```python
import numpy as np

kernel_name = "gpu_gemm_a2a_m2048_k2048_n2048_f32_silu"


ABSMAX_RTOL = 0.001

COMPUTE_DTYPE = "float32"
EPILOGUE = "silu"
IN_QUANT = None
OUT_QUANT = None
IN_DTYPES = {"x": "float32", "w_mat": "float32"}
WORLD_SIZE = 8

def setup_inputs():

    x = np.random.randn(2048, 2048).astype(np.float32)
    w_mat = (np.random.randn(2048, 2048) / np.sqrt(2048)).astype(np.float32)
    return {"x": x, "w_mat": w_mat}

def _epilogue(y):
    return (y / (1.0 + np.exp(-np.clip(y, -60.0, 60.0)))).astype(np.float32)

def reference(x, w_mat):
    y = (x @ w_mat).astype(np.float32)
    return _epilogue(y)
```

```cpp
#include <hip/hip_runtime.h>
#include <rccl/rccl.h>
#include <cstdio>

#define HIP_CHECK(expr) do { hipError_t _e = (expr); if (_e != hipSuccess) { \
    fprintf(stderr, "HIP error %s at %s:%d\n", hipGetErrorString(_e), __FILE__, __LINE__); \
    return; } } while (0)

typedef float f32x2 __attribute__((ext_vector_type(2)));
typedef float f32x4 __attribute__((ext_vector_type(4)));
typedef float f32x8 __attribute__((ext_vector_type(8)));
typedef unsigned u32x2 __attribute__((ext_vector_type(2)));
typedef unsigned u32x4 __attribute__((ext_vector_type(4)));
typedef _Float16 f16x8 __attribute__((ext_vector_type(8)));

constexpr int NR = 8;
constexpr int MS = 256;
constexpr int KK = 2048;
constexpr int NN = 2048;
constexpr int NS = 256;
constexpr int NWG = 256;
constexpr int NTHR = 512;
constexpr unsigned X_BYTES = MS * KK * 4;
constexpr unsigned W_BYTES = KK * NN * 4;
constexpr unsigned OUT_BYTES = NR * MS * NS * 4;
constexpr unsigned FLAG_OFF = 0;
constexpr unsigned CNT_OFF = NWG * 128;
constexpr unsigned TMO_OFF = 2 * NWG * 128;
constexpr unsigned WS_NEED = TMO_OFF + 128;
constexpr unsigned long long DEADLINE_TICKS = 50000000ull;

struct Params {
    const float* x;
    const float* w;
    void* out[NR];
    void* ws[NR];
    int rank;
    int pad;
};

__device__ __forceinline__ f16x8 cvt8(f32x4 a, f32x4 b) {
    f32x8 v = {a[0], a[1], a[2], a[3], b[0], b[1], b[2], b[3]};
    return __builtin_convertvector(v, f16x8);
}

__device__ __forceinline__ float silu(float y) {
    float c = fminf(fmaxf(y, -60.0f), 60.0f);
    return y / (1.0f + __expf(-c));
}

__global__ void __launch_bounds__(NTHR, 2) gemm_a2a(Params p) {
    __shared__ __attribute__((aligned(16))) float red[2][8][16 * 32];

    const int b = blockIdx.x;
    const int peer = b & 7;
    const int idx = b >> 3;
    const int mi = idx >> 3;
    const int ni = idx & 7;
    const int tid = threadIdx.x;
    const int wave = __builtin_amdgcn_readfirstlane(tid >> 6);
    const int lane = tid & 63;
    const int c = lane & 15, g = lane >> 4;
    const int rank = p.rank;

    char* myws = (char*)p.ws[rank];
    unsigned* cnt = (unsigned*)(myws + CNT_OFF + b * 128);
    unsigned call = __hip_atomic_load(cnt, __ATOMIC_RELAXED, __HIP_MEMORY_SCOPE_AGENT);

    const __amdgpu_buffer_rsrc_t rx = __builtin_amdgcn_make_buffer_rsrc((void*)p.x, 0, X_BYTES, 0x00020000);
    const __amdgpu_buffer_rsrc_t rw = __builtin_amdgcn_make_buffer_rsrc((void*)p.w, 0, W_BYTES, 0x00020000);
    const __amdgpu_buffer_rsrc_t ro = __builtin_amdgcn_make_buffer_rsrc(p.out[peer], 0, OUT_BYTES, 0x00020000);

    const int kb = wave * 256;
    const unsigned wv = (unsigned)((4 * g) * NN + 2 * c) * 4u;
    const unsigned ws0 = (unsigned)(kb * NN + peer * NS + ni * 32) * 4u;
    f32x2 wr[8][8];
#pragma unroll
    for (int s = 0; s < 8; ++s)
#pragma unroll
        for (int j = 0; j < 8; ++j) {
            const int krow = 32 * s + (j < 4 ? j : 12 + j);
            wr[s][j] = __builtin_bit_cast(f32x2, __builtin_amdgcn_raw_buffer_load_b64(rw, wv, ws0 + (unsigned)krow * NN * 4u, 0));
        }
    const unsigned xv = (unsigned)(c * KK + 4 * g) * 4u;
    const unsigned xs0 = (unsigned)(mi * 64 * KK + kb) * 4u;
    u32x4 xr[2][8][2];
    auto load_x = [&](int buf, int ch) {
#pragma unroll
        for (int s = 0; s < 8; ++s) {
            xr[buf][s][0] = __builtin_amdgcn_raw_buffer_load_b128(rx, xv + s * 128, xs0 + (unsigned)ch * 16u * KK * 4u, 0);
            xr[buf][s][1] = __builtin_amdgcn_raw_buffer_load_b128(rx, xv + s * 128 + 64, xs0 + (unsigned)ch * 16u * KK * 4u, 0);
        }
    };
    load_x(0, 0);
    __builtin_amdgcn_sched_barrier(0);

    f16x8 bf[8][2];
#pragma unroll
    for (int s = 0; s < 8; ++s)
#pragma unroll
        for (int q = 0; q < 2; ++q) {
            f32x8 v;
#pragma unroll
            for (int j = 0; j < 8; ++j) v[j] = wr[s][j][q];
            bf[s][q] = __builtin_convertvector(v, f16x8);
        }
    __builtin_amdgcn_sched_barrier(0);
    load_x(1, 1);
    __builtin_amdgcn_sched_barrier(0);

    const int orow = 2 * wave + ((lane >> 3) & 1);
    const int ocol = 4 * (lane & 7);
    const unsigned ov = lane < 16 ? (unsigned)(orow * NS + ocol) * 4u : 0x80000000u;
    const unsigned os0 = (unsigned)((rank * MS + mi * 64) * NS + ni * 32) * 4u;

#pragma unroll
    for (int ch = 0; ch < 4; ++ch) {
        const int buf = ch & 1;
        f32x4 acc0 = {0.f, 0.f, 0.f, 0.f}, acc1 = {0.f, 0.f, 0.f, 0.f};
#pragma unroll
        for (int s = 0; s < 8; ++s) {
            f16x8 af = cvt8(__builtin_bit_cast(f32x4, xr[buf][s][0]), __builtin_bit_cast(f32x4, xr[buf][s][1]));
            acc0 = __builtin_amdgcn_mfma_f32_16x16x32_f16(af, bf[s][0], acc0, 0, 0, 0);
            acc1 = __builtin_amdgcn_mfma_f32_16x16x32_f16(af, bf[s][1], acc1, 0, 0, 0);
        }
        __builtin_amdgcn_sched_barrier(0);
        if (ch + 2 < 4) load_x(buf, ch + 2);
        __builtin_amdgcn_sched_barrier(0);
        float* rp = &red[buf][wave][0];
#pragma unroll
        for (int i = 0; i < 4; ++i) {
            f32x2 v = {acc0[i], acc1[i]};
            *(f32x2*)&rp[(4 * g + i) * 32 + 2 * c] = v;
        }
        asm volatile("s_waitcnt lgkmcnt(0)" ::: "memory");
        __builtin_amdgcn_s_barrier();
        asm volatile("" ::: "memory");
        {
            f32x4 sum = *(const f32x4*)&red[buf][0][orow * 32 + ocol];
#pragma unroll
            for (int w = 1; w < 8; ++w) sum += *(const f32x4*)&red[buf][w][orow * 32 + ocol];
            f32x4 o = {silu(sum[0]), silu(sum[1]), silu(sum[2]), silu(sum[3])};
            __builtin_amdgcn_raw_buffer_store_b128(__builtin_bit_cast(u32x4, o), ro, ov,
                                                   os0 + (unsigned)ch * 16u * NS * 4u, 17);
        }
        __builtin_amdgcn_sched_barrier(0);
    }

    asm volatile("s_waitcnt vmcnt(0)" ::: "memory");
    __syncthreads();
    asm volatile("" : "+v"(call));
    call += 1;
    if (tid == 0) {
        if (peer != rank) {
            unsigned* flag = (unsigned*)((char*)p.ws[peer] + FLAG_OFF + (rank * 32 + idx) * 128);
            __hip_atomic_store(flag, call, __ATOMIC_RELAXED, __HIP_MEMORY_SCOPE_SYSTEM);
        }
        __hip_atomic_store(cnt, call, __ATOMIC_RELAXED, __HIP_MEMORY_SCOPE_AGENT);
        if ((b >> 5) != rank) {
            unsigned* flag = (unsigned*)(myws + FLAG_OFF + b * 128);
            unsigned* tmo = (unsigned*)(myws + TMO_OFF);
            unsigned long long t0 = __builtin_amdgcn_s_memrealtime();
            while (__hip_atomic_load(flag, __ATOMIC_RELAXED, __HIP_MEMORY_SCOPE_SYSTEM) < call) {
                if (__hip_atomic_load(tmo, __ATOMIC_RELAXED, __HIP_MEMORY_SCOPE_AGENT) == call ||
                    __builtin_amdgcn_s_memrealtime() - t0 > DEADLINE_TICKS) {
                    __hip_atomic_store(tmo, call, __ATOMIC_RELAXED, __HIP_MEMORY_SCOPE_AGENT);
                    unsigned* o = (unsigned*)p.out[rank] + (size_t)rank * MS * NS;
                    __hip_atomic_store(o, 0x7fc00000u, __ATOMIC_RELAXED, __HIP_MEMORY_SCOPE_SYSTEM);
                    break;
                }
                __builtin_amdgcn_s_sleep(1);
            }
        }
    }
}

extern "C" void kernel_launch(int rank, int nranks, ncclComm_t comm,
                              hipStream_t stream,
                              void* const* d_in, const int* in_sizes, int n_in,
                              void* d_out, int out_size,
                              const int* in_dtypes, int out_dtype,
                              void* const* d_out_all,
                              void* const* d_ws_all, size_t ws_bytes) {
    if (nranks != NR || (unsigned)rank >= (unsigned)NR || n_in != 2 || in_sizes[0] != MS * KK ||
        in_sizes[1] != KK * NN || out_size != NR * MS * NS || ws_bytes < WS_NEED) {
        fprintf(stderr, "gemm_a2a: unexpected problem shape\n");
        return;
    }
    Params p;
    p.x = (const float*)d_in[0];
    p.w = (const float*)d_in[1];
    for (int i = 0; i < NR; ++i) {
        p.out[i] = d_out_all[i];
        p.ws[i] = d_ws_all[i];
    }
    p.rank = rank;
    p.pad = 0;
    hipLaunchKernelGGL(gemm_a2a, dim3(NWG), dim3(NTHR), 0, stream, p);
    HIP_CHECK(hipGetLastError());
}
```

```cpp
#include <hip/hip_runtime.h>
#include <rccl/rccl.h>
#include <cstdio>

#define HIP_CHECK(expr) do { hipError_t _e = (expr); if (_e != hipSuccess) { \
    fprintf(stderr, "HIP error %s at %s:%d\n", hipGetErrorString(_e), __FILE__, __LINE__); \
    return; } } while (0)

typedef float f32x2 __attribute__((ext_vector_type(2)));
typedef _Float16 f16x2 __attribute__((ext_vector_type(2)));
typedef float f32x4 __attribute__((ext_vector_type(4)));
typedef float f32x8 __attribute__((ext_vector_type(8)));
typedef unsigned u32x4 __attribute__((ext_vector_type(4)));
typedef _Float16 f16x8 __attribute__((ext_vector_type(8)));
#define LDS_AS __attribute__((address_space(3)))
typedef __attribute__((address_space(1))) unsigned gu32;

constexpr int NR = 8;
constexpr int MS = 256;
constexpr int KK = 2048;
constexpr int NN = 2048;
constexpr int NS = 256;
constexpr int NWG = 256;
constexpr int NAUX = 2;
constexpr int NCW = 4;
constexpr int NTHR = (NCW + NAUX) * 64;
constexpr unsigned X_BYTES = MS * KK * 4;
constexpr unsigned W_BYTES = KK * NN * 4;
constexpr unsigned OUT_BYTES = NR * MS * NS * 4;
constexpr unsigned FLAG_OFF = 0;
constexpr unsigned CNT_OFF = NWG * 128;
constexpr unsigned TMO_OFF = 2 * NWG * 128;
constexpr unsigned X16_OFF = 128 * 1024;
constexpr unsigned X16_BYTES = MS * KK * 2;
constexpr unsigned WS_NEED = X16_OFF + 2 * X16_BYTES;
static_assert(TMO_OFF + 128 <= X16_OFF, "workspace layout");
constexpr unsigned long long DEADLINE_TICKS = 50000000ull;
constexpr int AUX_SYS = 17;

constexpr int L_PART = 0;
constexpr int L_DUMMY = 32768;
constexpr int L_PROG = L_DUMMY + 1024;
constexpr int L_TOTAL = L_PROG + 64;

#define STAMP(k) do {} while (0)
#define STAMP_FLUSH() do {} while (0)
#define SB() __builtin_amdgcn_sched_barrier(0)
#define WDEPTH 16
#define NE 4
#define HSKIP 32
#define PXOWN 2
#define OWNHELP 0

struct Params {
    const float* x;
    const float* w;
    void* myws;
    int rank;
    int pad;
    void* out[NR];
    void* ws[NR];
};

__device__ __forceinline__ f16x8 cvt8(u32x4 a, u32x4 b) {
    f32x4 fa = __builtin_bit_cast(f32x4, a), fb = __builtin_bit_cast(f32x4, b);
    f32x8 v = {fa[0], fa[1], fa[2], fa[3], fb[0], fb[1], fb[2], fb[3]};
    return __builtin_convertvector(v, f16x8);
}

__device__ __forceinline__ unsigned cvt2(float a, float b) {
    const f32x2 v = {a, b};
    return __builtin_bit_cast(unsigned, __builtin_convertvector(v, f16x2));
}

__device__ __forceinline__ float silu(float y) {
    float c = fminf(fmaxf(y, -60.0f), 60.0f);
    return y * __builtin_amdgcn_rcpf(1.0f + __builtin_amdgcn_exp2f(c * -1.4426950408889634f));
}

__device__ __forceinline__ bool wait_words(const unsigned* word, unsigned call, unsigned* tmo) {
    const unsigned long long t0 = __builtin_amdgcn_s_memrealtime();
    for (unsigned it = 1;; ++it) {
        const unsigned v = __hip_atomic_load(word, __ATOMIC_RELAXED, __HIP_MEMORY_SCOPE_SYSTEM);
        if (__all(v >= call)) return true;
        if ((it & 63u) == 0u &&
            (__hip_atomic_load(tmo, __ATOMIC_RELAXED, __HIP_MEMORY_SCOPE_AGENT) == call ||
             __builtin_amdgcn_s_memrealtime() - t0 > DEADLINE_TICKS)) {
            __hip_atomic_store(tmo, call, __ATOMIC_RELAXED, __HIP_MEMORY_SCOPE_AGENT);
            return false;
        }
        __builtin_amdgcn_s_sleep(1);
    }
}

__device__ __forceinline__ void dma_discard(__amdgpu_buffer_rsrc_t r, unsigned voff, unsigned soff, unsigned ldst) {
    unsigned keep;
    asm volatile("s_mov_b32 %0, m0\n\ts_mov_b32 m0, %1\n\ts_nop 0\n\tbuffer_load_dwordx4 %2, %3, %4 offen lds\n\ts_mov_b32 m0, %0"
                 : "=&s"(keep) : "s"(ldst), "v"(voff), "s"(r), "s"(soff) : "memory");
}

__device__ __forceinline__ unsigned min4(u32x4 v, unsigned m) {
    return min(min(min(v[0], v[1]), m), min(v[2], v[3]));
}

__global__ void __launch_bounds__(NTHR, 2) gemm_a2a(Params p) {

    __shared__ __attribute__((aligned(16))) char lds[L_TOTAL];

    const int b = blockIdx.x;
    const int xj = b & 7;
    const int q = (b >> 3) & 7;
    const int mi = b >> 6;
    const int tid = threadIdx.x;
    const int wave = __builtin_amdgcn_readfirstlane(tid >> 6);
    const int lane = tid & 63;
    const int rank = p.rank;
    char* myws = (char*)p.myws;
    const bool own = q == 7;
    auto slot_peer = [&](int qq) { return (rank + 1 + (qq + xj) % 7) & 7; };
    auto slot_tile = [&](int qq) { return (int)((0x06542173u >> (4 * (xj == 7 ? 7 : qq))) & 7u); };
    const int peer = own ? rank : slot_peer(q);
    const int ni = own ? xj : slot_tile(q);
    const bool late = q >= NE;
    const bool helper = NE > 0 && late && !own;
    STAMP(0);

    unsigned* cnt = (unsigned*)(myws + CNT_OFF + b * 128);
    unsigned* tmo = (unsigned*)(myws + TMO_OFF);
    unsigned callv = __hip_atomic_load(cnt, __ATOMIC_RELAXED, __HIP_MEMORY_SCOPE_AGENT);
    volatile LDS_AS unsigned* prog = (volatile LDS_AS unsigned*)(lds + L_PROG);
    const __amdgpu_buffer_rsrc_t rx16 = __builtin_amdgcn_make_buffer_rsrc(myws + X16_OFF, 0, 2 * X16_BYTES, 0x00020000);

    if (wave >= NCW) {
        const int a = wave - NCW;
        __builtin_amdgcn_s_setprio(3);
        if (lane < NCW) prog[lane] = 0;
        asm volatile("s_waitcnt lgkmcnt(0)" ::: "memory");
        __builtin_amdgcn_s_barrier();
        const unsigned call = __builtin_amdgcn_readfirstlane(callv) + 1;
        auto zero_next = [&]() {
            const u32x4 z = {0u, 0u, 0u, 0u};
            const unsigned zs = ((call + 1) & 1u) * X16_BYTES + (unsigned)b * 4096u + (unsigned)a * (4096u / NAUX);
#pragma unroll
            for (int i = 0; i < 4 / NAUX; ++i) __builtin_amdgcn_raw_buffer_store_b128(z, rx16, lane * 16 + i * 1024, zs, AUX_SYS);
        };
        void* outp = p.out[peer];
        char* peerws = (char*)p.ws[peer];
        asm volatile("" : "+s"(outp), "+s"(peerws));
        const __amdgpu_buffer_rsrc_t ro = __builtin_amdgcn_make_buffer_rsrc(outp, 0, OUT_BYTES, 0x00020000);
        const unsigned ov = (unsigned)((lane >> 3) * NS + (lane & 7) * 4) * 4u;
        const unsigned os0 = (unsigned)((rank * MS + mi * 64 + a * 8) * NS + ni * 32) * 4u;
#pragma unroll 1
        for (int c = -1; c < 4; ++c) {
            while (!__all((int)prog[lane & (NCW - 1)] > c)) __builtin_amdgcn_s_sleep(1);
            asm volatile("" ::: "memory");
            const char* pp = lds + L_PART + (c & 3) * 8192 + a * 1024 + lane * 16;
            f32x4 s0 = *(const f32x4*)(pp), s1 = *(const f32x4*)(pp + 2048), s2 = *(const f32x4*)(pp + 4096), s3 = *(const f32x4*)(pp + 6144);
            f32x4 sum = (s0 + s1) + (s2 + s3);
            f32x4 o = {silu(sum[0]), silu(sum[1]), silu(sum[2]), silu(sum[3])};
            __builtin_amdgcn_raw_buffer_store_b128(__builtin_bit_cast(u32x4, o), ro, c < 0 ? 0x80000000u : ov,
                                                   os0 + (unsigned)(c & 3) * 16u * NS * 4u, AUX_SYS);
        }
        STAMP(6);
        asm volatile("s_waitcnt vmcnt(0)" ::: "memory");
        STAMP(7);
        if (lane == 0) {
            if (peer != rank) {
                gu32* flag = (gu32*)(peerws + FLAG_OFF + (rank * 32 + mi * 8 + ni) * 128) + a;
                __hip_atomic_store(flag, call, __ATOMIC_RELAXED, __HIP_MEMORY_SCOPE_SYSTEM);
            }
            if (a == 0) __hip_atomic_store(cnt, call, __ATOMIC_RELAXED, __HIP_MEMORY_SCOPE_AGENT);
        }
        zero_next();
        STAMP_FLUSH();
        return;
    }

    const int w = wave;
    const int kb = w * 512;
    const unsigned ldummy = (unsigned)(size_t)(LDS_AS char*)(lds + L_DUMMY);
    const __amdgpu_buffer_rsrc_t rw = __builtin_amdgcn_make_buffer_rsrc((void*)p.w, 0, W_BYTES, 0x00020000);
    {
        if (own) {
            const __amdgpu_buffer_rsrc_t rx = __builtin_amdgcn_make_buffer_rsrc((void*)p.x, 0, X_BYTES, 0x00020000);
            const unsigned xv = (unsigned)((lane >> 3) * KK + (lane & 7) * 4) * 4u;
            const unsigned xs = (unsigned)(mi * 64 * KK + (xj * 4 + w) * 64) * 4u;
            u32x4 xr[8][2];
#pragma unroll
            for (int i = 0; i < 8; ++i) {
                xr[i][0] = __builtin_amdgcn_raw_buffer_load_b128(rx, xv, xs + (unsigned)i * 8u * KK * 4u, 0);
                xr[i][1] = __builtin_amdgcn_raw_buffer_load_b128(rx, xv + 128, xs + (unsigned)i * 8u * KK * 4u, 0);
            }
            SB();
            char* tr = lds + L_PART + w * 2048;
            const int m = lane >> 3, h = lane & 7;
            const unsigned par = (__builtin_amdgcn_readfirstlane(callv) + 1) & 1u;
            const unsigned sv = (unsigned)((m & 1) * 1024 + (m >> 1) * 256 + h * 16);
            const unsigned ss = par * X16_BYTES + (unsigned)(mi * 4 * 65536 + (xj * 4 + w) * 2048);
#pragma unroll
            for (int i = 0; i < 8; ++i) {
                u32x4 v = __builtin_bit_cast(u32x4, cvt8(xr[i][0], xr[i][1]));
#pragma unroll
                for (int d = 0; d < 4; ++d) v[d] = v[d] == 0u ? 0x00008000u : v[d];
                char* t = tr + (i >> 1) * 8192 + (i & 1) * 1024;
                *(u32x4*)(t + (h * 8 + (m ^ h)) * 16) = v;
                v = *(const u32x4*)(t + (m * 8 + (h ^ m)) * 16);
                __builtin_amdgcn_raw_buffer_store_b128(v, rx16, sv + (i & 1) * 128, ss + (i >> 1) * 65536, AUX_SYS);
                if (i & 1) SB();
            }
#pragma unroll 1
            for (int i = 0; i < PXOWN * 16; ++i)
                dma_discard(rx16, (unsigned)lane * 16u, par * X16_BYTES + (unsigned)(mi * 4 * 65536 + (i >> 4) * 65536 + w * 16384 + (i & 15) * 1024), ldummy);
        } else if (helper) {
            constexpr int NE1 = NE > 0 ? NE : 1, NH = (7 - NE) * 4, NGRP = (512 - HSKIP) / 8;
            const int hid = (q - NE) * 4 + mi;
            unsigned hv[NE1];
#pragma unroll
            for (int i = 0; i < NE1; ++i) {
                const int sg = i * 8 + (lane >> 3), row = sg / NE1, e = sg - row * NE1;
                hv[i] = (unsigned)(row * NN + slot_peer(e) * NS + slot_tile(e) * 32 + (lane & 7) * 4) * 4u;
            }
#pragma unroll 1
            for (int g = hid; g < NGRP; g += NH) {
#pragma unroll
                for (int i = 0; i < NE1; ++i) dma_discard(rw, hv[i], (unsigned)((kb + HSKIP + g * 8) * NN) * 4u, ldummy);
            }
        }
    }
    STAMP(1);
    const int rot = late ? 2 * mi : 0;
    const unsigned wv = (unsigned)((lane >> 3) * 4 * NN + (lane & 7) * 4) * 4u;
    const unsigned ws0 = (unsigned)(kb * NN + peer * NS + ni * 32) * 4u;
    const unsigned av = (unsigned)lane * 16u;
    u32x4 wr[64];
    u32x4 bf[8][2][2];
    u32x4 ar[16];
    unsigned call = 0, as0 = 0;
    auto load_w = [&](int t) {
        const int s = t >> 3, j = t & 7;
        wr[t] = __builtin_amdgcn_raw_buffer_load_b128(rw, wv, ws0 + (unsigned)(((s + rot) & 7) * 64 + (j < 4 ? j : 28 + j)) * NN * 4u, 0);
    };
    auto load_a = [&](int t, unsigned c) {
        ar[t] = __builtin_amdgcn_raw_buffer_load_b128(rx16, (av | ((c >> 2) << 31)) + (t & 1) * 1024, as0 + (c & 3u) * 65536u + (unsigned)(((t >> 1) + rot) & 7) * 2048u, AUX_SYS);
    };
#pragma unroll
    for (int t = 0; t < WDEPTH; ++t) {
        load_w(t);
        if (t == 3) { SB(); __builtin_amdgcn_s_barrier(); SB(); }
    }
    SB();
#pragma unroll
    for (int t = 0; t < 64; t += 2) {
        const int s = t >> 3, d = (t & 7) >> 1;
        const f32x4 lo = __builtin_bit_cast(f32x4, wr[t]), hi = __builtin_bit_cast(f32x4, wr[t + 1]);
#pragma unroll
        for (int q0 = 0; q0 < 2; ++q0) {
            const unsigned r0 = cvt2(lo[q0], hi[q0]), r1 = cvt2(lo[q0 + 2], hi[q0 + 2]);
            unsigned e0 = (unsigned)__builtin_amdgcn_update_dpp((int)r0, (int)r1, 0x128, 0xf, 0xc, false);
            unsigned e1 = (unsigned)__builtin_amdgcn_update_dpp((int)r1, (int)r0, 0x128, 0xf, 0x3, false);
            asm volatile("" : "+v"(e0), "+v"(e1));
            bf[s][0][q0][d] = e0;
            bf[s][1][q0][d] = e1;
        }
        if (t == 0) {
            call = __builtin_amdgcn_readfirstlane(callv) + 1;
            as0 = (call & 1u) * X16_BYTES + (unsigned)(mi * 4 * 65536 + w * 16384);
        }
        SB();
#pragma unroll
        for (int u = 0; u < 2; ++u) {
            const int n = t + u + WDEPTH;
            if (n < 64) load_w(n);
            else if (n < 80) load_a(n - 64, 0);
        }
        SB();
    }
#pragma unroll
    for (int n = WDEPTH; n < 16; ++n) load_a(n, 0);
    STAMP(3);

    {
        const unsigned pofs = L_PART + w * 2048 + (unsigned)((lane >> 4) * 512 + (lane & 7) * 16 + ((lane >> 3) & 1) * 8);
        float bad = 0.0f;
        unsigned long long t0 = 0;
        unsigned c = 0, have = 0;
#pragma unroll 1
        while (c < 4) {
            const unsigned cn = have != c ? c : c + 1;
            unsigned mn = 0xffffffffu;
            f32x4 acc0 = {bad, bad, bad, bad}, acc1 = acc0;
#pragma unroll
            for (int t = 0; t < 16; ++t) {
                acc0 = __builtin_amdgcn_mfma_f32_16x16x32_f16(__builtin_bit_cast(f16x8, ar[t]), __builtin_bit_cast(f16x8, bf[t >> 1][t & 1][0]), acc0, 0, 0, 0);
                acc1 = __builtin_amdgcn_mfma_f32_16x16x32_f16(__builtin_bit_cast(f16x8, ar[t]), __builtin_bit_cast(f16x8, bf[t >> 1][t & 1][1]), acc1, 0, 0, 0);
                mn = min4(ar[t], mn);
                asm volatile("" : "+v"(mn) :: "memory");
                SB();
                if (t < 15) { load_a(t, cn); SB(); }
            }
            if (have == c) {
                if (__builtin_expect(!__any(mn == 0u) || bad != 0.0f, 1)) {
                    volatile LDS_AS float* pp = (volatile LDS_AS float*)(lds + pofs + c * 8192u);
#pragma unroll
                    for (int i = 0; i < 4; ++i) { pp[i * 32] = acc0[i]; pp[i * 32 + 1] = acc1[i]; }
                    prog[w] = c + 1;
                    ++c;
                } else {
                    if (t0 == 0) t0 = __builtin_amdgcn_s_memrealtime();
                    if (__builtin_amdgcn_s_memrealtime() - t0 > DEADLINE_TICKS ||
                        __hip_atomic_load(tmo, __ATOMIC_RELAXED, __HIP_MEMORY_SCOPE_AGENT) == call) {
                        __hip_atomic_store(tmo, call, __ATOMIC_RELAXED, __HIP_MEMORY_SCOPE_AGENT);
                        bad = __builtin_nanf("");
                    }
                    __builtin_amdgcn_s_waitcnt(0x0F70);
                }
            }
            SB();
            load_a(15, cn);
            SB();
            have = cn;
        }
    }
    if (w == 0 && (b >> 5) != rank) {
        if (!wait_words((const unsigned*)(myws + FLAG_OFF + b * 128) + (lane & (NAUX - 1)), call, tmo)) {
            unsigned* o = (unsigned*)p.out[rank] + (size_t)(b >> 5) * MS * NS;
            if (lane == 0) __hip_atomic_store(o, 0x7fc00000u, __ATOMIC_RELAXED, __HIP_MEMORY_SCOPE_SYSTEM);
        }
        STAMP(7);
    }
    STAMP_FLUSH();
}

extern "C" void kernel_launch(int rank, int nranks, ncclComm_t comm,
                              hipStream_t stream,
                              void* const* d_in, const int* in_sizes, int n_in,
                              void* d_out, int out_size,
                              const int* in_dtypes, int out_dtype,
                              void* const* d_out_all,
                              void* const* d_ws_all, size_t ws_bytes) {
    if (nranks != NR || (unsigned)rank >= (unsigned)NR || n_in != 2 || in_sizes[0] != MS * KK ||
        in_sizes[1] != KK * NN || out_size != NR * MS * NS || ws_bytes < WS_NEED) {
        fprintf(stderr, "gemm_a2a: unexpected problem shape\n");
        return;
    }
    Params p;
    p.x = (const float*)d_in[0];
    p.w = (const float*)d_in[1];
    for (int i = 0; i < NR; ++i) {
        p.out[i] = d_out_all[i];
        p.ws[i] = d_ws_all[i];
    }
    p.myws = d_ws_all[rank];
    p.rank = rank;
    p.pad = 0;
    hipLaunchKernelGGL(gemm_a2a, dim3(NWG), dim3(NTHR), 0, stream, p);
    HIP_CHECK(hipGetLastError());
}
```

```cpp
#include <hip/hip_runtime.h>
#include <rccl/rccl.h>
#include <cstdio>
#include <cstdlib>
#include <cstring>

#define HIP_CHECK(expr) do { hipError_t _e = (expr); if (_e != hipSuccess) { \
    fprintf(stderr, "HIP error %s at %s:%d\n", hipGetErrorString(_e), __FILE__, __LINE__); \
    return; } } while (0)

typedef float f32x2 __attribute__((ext_vector_type(2)));
typedef _Float16 f16x2 __attribute__((ext_vector_type(2)));
typedef float f32x4 __attribute__((ext_vector_type(4)));
typedef float f32x8 __attribute__((ext_vector_type(8)));
typedef unsigned u32x4 __attribute__((ext_vector_type(4)));
typedef unsigned u32x16 __attribute__((ext_vector_type(16)));
typedef _Float16 f16x8 __attribute__((ext_vector_type(8)));
#define LDS_AS __attribute__((address_space(3)))
#define GLB __attribute__((address_space(1)))
typedef GLB unsigned gu32;

constexpr int NR = 8;
constexpr int MS = 256;
constexpr int KK = 2048;
constexpr int NN = 2048;
constexpr int NS = 256;
constexpr int NWG = 256;
constexpr int NCW = 4;
constexpr int NAUX = 2;
constexpr int NTHR = (NCW + NAUX) * 64;
constexpr unsigned X_BYTES = MS * KK * 4;
constexpr unsigned W_BYTES = KK * NN * 4;
constexpr unsigned OUT_BYTES = NR * MS * NS * 4;
constexpr int MAXF = 4;
constexpr int MAXCH = 8;
constexpr int NB = 8;
constexpr int NBF = 4;
constexpr int MAXH = 24;

constexpr unsigned FLAG_OFF = 0;
constexpr unsigned CNT_OFF = 64 * 128;
constexpr unsigned RDY_OFF = CNT_OFF + NWG * 128;
constexpr unsigned TMO_OFF = RDY_OFF + NWG * 128;
constexpr unsigned X16_OFF = 128 * 1024;
constexpr unsigned X16_BYTES = (16 + 7 * MAXF) * 65536;
constexpr unsigned WS_NEED = X16_OFF + 2 * X16_BYTES;
static_assert(TMO_OFF + 128 <= X16_OFF, "workspace layout");
constexpr unsigned long long DEADLINE_TICKS = 50000000ull;
constexpr int AUX_SYS = 17;
constexpr unsigned MARK = 0xffffffffu;
constexpr unsigned OOB = 0x80000000u;

constexpr int L_DUMMY = 0;
constexpr int L_PROG = 1024;
constexpr int L_PART = 2048;
constexpr int L_CH = NCW * 2048;
constexpr int L_TOTAL = L_PART + MAXCH * L_CH;

#define STAMP(k) do {} while (0)
#define STAMP_FLUSH() do {} while (0)
#define SB() __builtin_amdgcn_sched_barrier(0)
#define TL 4
#define SPEC 0

template <bool B> struct BoolC { static constexpr bool value = B; };

struct Slot {
    unsigned a;
    unsigned b;
    unsigned ord;
    unsigned rel;
    unsigned short h[MAXH];
};
static_assert(sizeof(Slot) == 64, "slot");
struct Params {
    const float* x;
    const float* w;
    void* myws;
    int rank;
    unsigned cfg;
    unsigned words;
    unsigned pad;
    unsigned long long col[8];
    void* out[NR];
    void* ws[NR];
    unsigned long long pad2[3];
    Slot slot[32];
};
static_assert(offsetof(Params, slot) % 64 == 0, "alignment");

__device__ __forceinline__ u32x4 cvt8(u32x4 a, u32x4 b) {
    f32x4 fa = __builtin_bit_cast(f32x4, a), fb = __builtin_bit_cast(f32x4, b);
    f32x8 v = {fa[0], fa[1], fa[2], fa[3], fb[0], fb[1], fb[2], fb[3]};
    return __builtin_bit_cast(u32x4, __builtin_convertvector(v, f16x8));
}

__device__ __forceinline__ unsigned cvt2(float a, float b) {
    const f32x2 v = {a, b};
    return __builtin_bit_cast(unsigned, __builtin_convertvector(v, f16x2));
}

__device__ __forceinline__ float add1(float a, float b) {
    float r;
    asm("v_add_f32 %0, %1, %2" : "=v"(r) : "v"(a), "v"(b));
    return r;
}

__device__ __forceinline__ float silu(float y) {
    const float c = fminf(fmaxf(y, -60.0f), 60.0f);
    const float t = __builtin_amdgcn_rcpf(1.0f + __builtin_amdgcn_exp2f(c * -1.4426950408889634f));
    float r;
    asm("s_nop 0\n\tv_mul_f32 %0, %1, %2" : "=v"(r) : "v"(y), "v"(t));
    return r;
}

template <int NAP>
__device__ __forceinline__ bool wait_words(gu32* word, unsigned call, gu32* tmo) {
    const unsigned long long t0 = __builtin_amdgcn_s_memrealtime();
    for (unsigned it = 1;; ++it) {
        const unsigned v = __hip_atomic_load(word, __ATOMIC_RELAXED, __HIP_MEMORY_SCOPE_SYSTEM);
        if (__all(v >= call)) return true;
        if ((it & 63u) == 0u &&
            (__hip_atomic_load(tmo, __ATOMIC_RELAXED, __HIP_MEMORY_SCOPE_AGENT) == call ||
             __builtin_amdgcn_s_memrealtime() - t0 > DEADLINE_TICKS)) {
            __hip_atomic_store(tmo, call, __ATOMIC_RELAXED, __HIP_MEMORY_SCOPE_AGENT);
            return false;
        }
        __builtin_amdgcn_s_sleep(NAP);
    }
}

__device__ __forceinline__ void store_sys(u32x4 v, __amdgpu_buffer_rsrc_t r, unsigned voff, unsigned soff) {
    __builtin_amdgcn_raw_buffer_store_b128(v, r, voff, soff, AUX_SYS);
    asm volatile("s_nop 1" :: "v"(v) : "memory");
}

__device__ __forceinline__ void touch(__amdgpu_buffer_rsrc_t r, unsigned voff, unsigned soff, unsigned ldst) {
    unsigned keep;
    asm volatile("s_mov_b32 %0, m0\n\ts_mov_b32 m0, %1\n\ts_nop 0\n\tbuffer_load_dword %2, %3, %4 offen lds\n\ts_mov_b32 m0, %0"
                 : "=&s"(keep) : "s"(ldst), "v"(voff), "s"(r), "s"(soff) : "memory");
}

__device__ __forceinline__ unsigned min4(u32x4 v, unsigned m) {
    return min(min(min(min(m, v[0]), v[1]), v[2]), v[3]);
}

constexpr unsigned TILE_OF_Q = 0x06542173u;
constexpr unsigned Q_OF_TILE = 0x16540327u;

__global__ void __launch_bounds__(NTHR, 2) gemm_a2a(Params p) {
    __shared__ __attribute__((aligned(16))) char lds[L_TOTAL];

    const int b = blockIdx.x;
    const int xj = b & 7;
    const int sl = b >> 3;
    const int tid = threadIdx.x;
    const int wave = __builtin_amdgcn_readfirstlane(tid >> 6);
    const int lane = tid & 63;
    int rank = p.rank;
    GLB char* myws = (GLB char*)p.myws;
    const float* px = p.x;
    const float* pw = p.w;
    unsigned cfg = p.cfg;
    unsigned long long col = p.col[xj];
    u32x16 sd = *(const u32x16*)&p.slot[sl];
    asm volatile("" : "+s"(rank), "+s"(myws), "+s"(px), "+s"(pw), "+s"(cfg), "+s"(col), "+s"(sd));
    const unsigned da = sd[0], db = sd[1], ord = sd[2];
    const int q = da & 7;
    const int tcol = (int)(col >> (8 * q)) & 63;
    const int peer = tcol >> 3;
    const int ni = tcol & 7;
    const int c0 = (da >> 7) & 63, cs = (da >> 13) & 15, n = (da >> 17) & 15;
    const bool own = q == 7;
    const bool fwd = (da >> 22) & 1;
    const int rch = cfg & 31;
    STAMP(0);

    gu32* cnt = (gu32*)(myws + CNT_OFF + b * 128);
    gu32* tmo = (gu32*)(myws + TMO_OFF);
    unsigned callv = __hip_atomic_load(cnt, __ATOMIC_RELAXED, __HIP_MEMORY_SCOPE_AGENT);
    volatile LDS_AS unsigned* prog = (volatile LDS_AS unsigned*)(lds + L_PROG);
    const __amdgpu_buffer_rsrc_t rx16 = __builtin_amdgcn_make_buffer_rsrc((char*)myws + X16_OFF, 0, 2 * X16_BYTES, 0x00020000);

    if (__builtin_expect(wave >= NCW, 0)) {
        const int a = wave - NCW;
        unsigned rel = sd[3];
        __builtin_amdgcn_s_setprio(3);
        if (lane < NCW) prog[lane] = 0;
        const unsigned ov = (unsigned)((a * 8 + (lane >> 3)) * NS + ni * 32 + (lane & 7) * 4) * 4u;
        const bool marked = (da >> 23) & 1;
        const int src = (2 * rank - peer) & 7;
        if (marked) {
            const __amdgpu_buffer_rsrc_t rm = __builtin_amdgcn_make_buffer_rsrc(p.out[rank], 0, OUT_BYTES, 0x00020000);
            const u32x4 mk = {MARK, MARK, MARK, MARK};
#pragma unroll 1
            for (int c = 0; c < n; ++c) store_sys(mk, rm, ov, (unsigned)((src * MS + (c0 + c * cs) * 16) * NS) * 4u);
        }
        asm volatile("s_waitcnt lgkmcnt(0)" ::: "memory");
        __builtin_amdgcn_s_barrier();
        const unsigned call = __builtin_amdgcn_readfirstlane(callv) + 1;
        if (marked) {
            asm volatile("s_waitcnt vmcnt(0)" ::: "memory");
            if (lane == 0) {
                __hip_atomic_store((gu32*)((char*)p.ws[src] + RDY_OFF + b * 128) + a, call, __ATOMIC_RELAXED, __HIP_MEMORY_SCOPE_SYSTEM);
                __hip_atomic_store((gu32*)(myws + RDY_OFF + b * 128) + 2 + a, call, __ATOMIC_RELAXED, __HIP_MEMORY_SCOPE_SYSTEM);
            }
            __builtin_amdgcn_s_sleep(32);
            if (!wait_words<16>((gu32*)(myws + RDY_OFF + b * 128) + a, call, tmo)) rel = 0;
        }
        void* outp = p.out[peer];
        char* peerws = (char*)p.ws[peer];
        asm volatile("" : "+s"(outp), "+s"(peerws));
        const __amdgpu_buffer_rsrc_t ro = __builtin_amdgcn_make_buffer_rsrc(outp, 0, OUT_BYTES, 0x00020000);
        u32x4 o1 = {0u, 0u, 0u, 0u}, o2 = o1;
        unsigned os1 = 0, os2 = 0;
#pragma unroll 1
        for (int c = -1; c < n; ++c) {
            const int id = c0 + c * cs;
            const int fi = id - 16, fj = fi / 7;
            const int row0 = id < 16 ? rank * MS + id * 16 : ((rank + 1 + fi - fj * 7) & 7) * MS + (rch + fj) * 16;
            unsigned os = (unsigned)(row0 * NS) * 4u;
            unsigned act = (unsigned)(((unsigned long long)rel << 4) >> (4 * (c + 1))) & 7u;
            unsigned pa = (unsigned)(L_PART + max(c, 0) * L_CH + a * 1024 + lane * 16);
            asm volatile("" : "+s"(os), "+s"(act), "+v"(pa));
            while (!__all((int)prog[lane & (NCW - 1)] > c)) __builtin_amdgcn_s_sleep(1);
            asm volatile("" ::: "memory");
            const f32x4* pp = (const f32x4*)(lds + pa);
            const f32x4 s0 = pp[0], s1 = pp[128], s2 = pp[256], s3 = pp[384];
            if (act & 4u) store_sys(o2, ro, ov, os2);
            if (act & 2u) store_sys(o1, ro, ov, os1);
            f32x4 o;
#pragma unroll
            for (int i = 0; i < 4; ++i) o[i] = silu(add1(add1(s0[i], s1[i]), add1(s2[i], s3[i])));
            o2 = o1; os2 = os1;
            o1 = __builtin_bit_cast(u32x4, o); os1 = os;
#pragma unroll
            for (int i = 0; i < 4; ++i) o1[i] = min(o1[i], MARK - 1u);
            if (act & 1u) store_sys(o1, ro, ov, os);
        }
        STAMP(6);
        const int zi = (int)((db >> 15) & 31) - 1, zn = (cfg >> 5) & 31;
        if (zi >= 0) {
            const u32x4 z = {0u, 0u, 0u, 0u};
            const unsigned zb = ((call + 1) & 1u) * X16_BYTES;
            if (xj < 7) {
                const int si = (peer - rank - 1) & 7;
#pragma unroll 1
                for (int u = zi * NAUX + a; u < (16 - rch) * 64; u += zn * NAUX)
                    store_sys(z, rx16, lane * 16, zb + (unsigned)((16 + (u >> 6) * 7 + si) * 64 + (u & 63)) * 1024u);
            }
#pragma unroll 1
            for (int u = (zi * 8 + xj) * NAUX + a; u < 1024; u += zn * 8 * NAUX)
                store_sys(z, rx16, lane * 16, zb + (unsigned)u * 1024u);
        }
        asm volatile("s_waitcnt vmcnt(0)" ::: "memory");
        STAMP(7);
        if (lane == 0) {
            if (!own && !marked) {
                gu32* flag = (gu32*)(peerws + FLAG_OFF + (rank * 8 + ni) * 128) + ((da >> 3) & 15) * NAUX + a;
                __hip_atomic_store(flag, call, __ATOMIC_RELAXED, __HIP_MEMORY_SCOPE_SYSTEM);
            }
            if (a == 0) __hip_atomic_store(cnt, call, __ATOMIC_RELAXED, __HIP_MEMORY_SCOPE_AGENT);
        }
        STAMP_FLUSH();
        return;
    }

    const int w = wave;
    const int kb = w * 512;
    const unsigned av = (unsigned)lane * 16u;
    const __amdgpu_buffer_rsrc_t rw = __builtin_amdgcn_make_buffer_rsrc((void*)pw, 0, W_BYTES, 0x00020000);
    if (__builtin_expect((da >> 21) & 0x19u, 0)) {
    if ((da >> 21) & 1) {
        const int rb0 = db & 31, rbs = (db >> 5) & 15, rbe = (db >> 9) & 63;
        const int ks = xj * 4 + w;
        const __amdgpu_buffer_rsrc_t rx = __builtin_amdgcn_make_buffer_rsrc((void*)px, 0, X_BYTES, 0x00020000);
        const unsigned xv = (unsigned)((lane >> 3) * KK + (lane & 7) * 4) * 4u;
        u32x4 xr[NB][2];
#pragma unroll
        for (int i = 0; i < NB; ++i) {
            const int rb = rb0 + i * rbs;
            const unsigned xs = (unsigned)((rb & 31) * 8 * KK + ks * 64) * 4u, v = rb < rbe ? xv : OOB;
            xr[i][0] = __builtin_amdgcn_raw_buffer_load_b128(rx, v, xs, 0);
            xr[i][1] = __builtin_amdgcn_raw_buffer_load_b128(rx, v + 128, xs, 0);
        }
        SB();
        char* tr = lds + L_PART + w * 2048;
        const int m = lane >> 3, h = lane & 7;
        const unsigned par = (__builtin_amdgcn_readfirstlane(callv) + 1) & 1u;
        const unsigned sv = (unsigned)((m & 1) * 1024 + (m >> 1) * 256 + h * 16);
        const unsigned ss = par * X16_BYTES + (unsigned)(ks * 2048);
#pragma unroll
        for (int i2 = 0; i2 < NB; i2 += 2) {
#pragma unroll
            for (int i = i2; i < i2 + 2; ++i) {
                u32x4 v = cvt8(xr[i][0], xr[i][1]);
#pragma unroll
                for (int d = 0; d < 4; ++d) v[d] = v[d] == 0u ? 0x00008000u : v[d];
                char* t = tr + (i >> 1) * L_CH + (i & 1) * 1024;
                *(u32x4*)(t + (h * 8 + (m ^ h)) * 16) = v;
                xr[i][0] = *(const u32x4*)(t + (m * 8 + (h ^ m)) * 16);
            }
#pragma unroll
            for (int i = i2; i < i2 + 2; ++i) {
                const int rb = rb0 + i * rbs;
                store_sys(xr[i][0], rx16, rb < rbe ? sv : OOB, ss + (unsigned)(((rb >> 1) & 15) * 65536 + (rb & 1) * 128));
            }
            SB();
            if (rb0 + (i2 + 2) * rbs >= rbe) break;
        }
        if (fwd) {
#pragma unroll 1
            for (int d = 0; d < 7; ++d) {
                const int dd = (d + xj) % 7, dst = (rank + 1 + dd) & 7;
                const __amdgpu_buffer_rsrc_t rd = __builtin_amdgcn_make_buffer_rsrc((char*)p.ws[dst] + X16_OFF, 0, 2 * X16_BYTES, 0x00020000);
#pragma unroll
                for (int i = 0; i < NBF; ++i) {
                    const int rb = rb0 + i * rbs;
                    if (rb < rbe) store_sys(xr[i][0], rd, sv, ss + (unsigned)((16 + ((rb >> 1) - rch) * 7 + 6 - dd) * 65536 + (rb & 1) * 128));
                }
            }
        }
    }
    {
        const int ng = (da >> 24) & 3;
#pragma unroll
        for (int g = 0; g < MAXH / 8; ++g) {
            if (g < ng) {
                unsigned h0 = sd[4 + 4 * g], h1 = sd[5 + 4 * g], h2 = sd[6 + 4 * g], h3 = sd[7 + 4 * g];
                asm volatile("" : "+s"(h0), "+s"(h1), "+s"(h2), "+s"(h3));
                const unsigned d01 = (lane & 16) ? h1 : h0, d23 = (lane & 16) ? h3 : h2;
                const unsigned e = (((lane & 32) ? d23 : d01) >> (2 * (lane & 8))) & 0xffffu;
                const unsigned none = e == 0xffffu ? OOB : 0u;
                if (h0 & 0x8000u) {
                    unsigned cv;
                    asm volatile("s_load_dword %0, %1, 0x0\n\ts_waitcnt lgkmcnt(0)" : "=s"(cv) : "s"(cnt) : "memory");
                    touch(rx16, (((e >> 4) & 63u) * 65536u + (e & 15u) * 1024u + (lane & 7) * 128u) | none,
                          ((cv + 1) & 1u) * X16_BYTES + (unsigned)w * 16384u, L_DUMMY);
                } else {
                    touch(rw, (((e & 63u) * 8u + (lane & 7)) * (NN * 4u) + ((unsigned)(col >> (8 * ((e >> 6) & 7u))) & 63u) * 128u) | none,
                          (unsigned)kb * (NN * 4u), L_DUMMY);
                }
            }
        }
    }
    }
    STAMP(1);
    const unsigned wv = (unsigned)((lane >> 3) * 4 * NN + (lane & 7) * 4) * 4u;
    const unsigned ws0 = (unsigned)(kb * NN + tcol * 32) * 4u;
    u32x4 wr[64];
    u32x4 bf[8][2][2];
    u32x4 ar[16];
    unsigned call = 0, as0 = 0;
    const unsigned astep = (unsigned)cs * 65536u;
    auto step = [&](int s) { return (int)((ord >> (4 * s)) & 7u); };
    auto load_w = [&](int t) {
        const int s = t >> 3, j = t & 7;
        wr[t] = __builtin_amdgcn_raw_buffer_load_b128(rw, wv, ws0 + (unsigned)(step(s) * 64 + (j < 4 ? j : 28 + j)) * NN * 4u, 0);
    };
    auto load_a = [&](int t, unsigned c) {
        ar[t] = __builtin_amdgcn_raw_buffer_load_b128(rx16, av + (t & 1) * 1024, as0 + c * astep + (unsigned)step(t >> 1) * 2048u, AUX_SYS);
    };
#pragma unroll
    for (int t = 0; t < 16; ++t) {
        load_w(t);
        if (t == 3) { SB(); __builtin_amdgcn_s_barrier(); SB(); }
    }
    SB();
#pragma unroll
    for (int t = 0; t < 64; t += 2) {
        const int s = t >> 3, d = (t & 7) >> 1;
        const f32x4 lo = __builtin_bit_cast(f32x4, wr[t]), hi = __builtin_bit_cast(f32x4, wr[t + 1]);
#pragma unroll
        for (int q0 = 0; q0 < 2; ++q0) {
            const unsigned r0 = cvt2(lo[q0], hi[q0]), r1 = cvt2(lo[q0 + 2], hi[q0 + 2]);
            unsigned e0 = (unsigned)__builtin_amdgcn_update_dpp((int)r0, (int)r1, 0x128, 0xf, 0xc, false);
            unsigned e1 = (unsigned)__builtin_amdgcn_update_dpp((int)r1, (int)r0, 0x128, 0xf, 0x3, false);
            asm volatile("" : "+v"(e0), "+v"(e1));
            bf[s][0][q0][d] = e0;
            bf[s][1][q0][d] = e1;
        }
        if (t == 0) {
            call = __builtin_amdgcn_readfirstlane(callv) + 1;
            as0 = (call & 1u) * X16_BYTES + (unsigned)(c0 * 65536 + w * 16384);
        }
        SB();
#pragma unroll
        for (int u = 0; u < 2; ++u) {
            const int nn = t + u + 16;
            if (nn < 64) load_w(nn);
            else load_a(nn - 64, 0);
        }
        SB();
    }
    STAMP(3);

    {
        const unsigned pofs = L_PART + w * 2048 + (unsigned)((lane >> 4) * 512 + (lane & 7) * 16 + ((lane >> 3) & 1) * 8);
        float bad = 0.0f;
        unsigned long long t0 = 0;
        const unsigned last = (unsigned)n - 1u;
        unsigned c = 0, have = 0;
        auto pass = [&](auto more, unsigned cn) {
            unsigned mn = 0xffffffffu;
            f32x4 acc0 = {bad, bad, bad, bad}, acc1 = acc0;
#pragma unroll
            for (int t = 0; t < 16; ++t) {
                acc0 = __builtin_amdgcn_mfma_f32_16x16x32_f16(__builtin_bit_cast(f16x8, ar[t]), __builtin_bit_cast(f16x8, bf[t >> 1][t & 1][0]), acc0, 0, 0, 0);
                acc1 = __builtin_amdgcn_mfma_f32_16x16x32_f16(__builtin_bit_cast(f16x8, ar[t]), __builtin_bit_cast(f16x8, bf[t >> 1][t & 1][1]), acc1, 0, 0, 0);
                mn = min4(ar[t], mn);
                asm volatile("" : "+v"(mn) :: "memory");
                SB();
                if (decltype(more)::value && t < 16 - TL) { load_a(t, cn); SB(); }
            }
            const bool done = have == c && (!__any(mn == 0u) || bad != 0.0f);
            if (done) {
                volatile LDS_AS float* pp = (volatile LDS_AS float*)(lds + pofs + c * (unsigned)L_CH);
#pragma unroll
                for (int i = 0; i < 4; ++i) { pp[i * 32] = acc0[i]; pp[i * 32 + 1] = acc1[i]; }
                if (c + 1 != last || TL == 0) prog[w] = c + 1;
            }
            SB();
            if (decltype(more)::value && TL > 0) {
#pragma unroll
                for (int t = 16 - TL; t < 16; ++t) load_a(t, cn);
                SB();
                if (done && c + 1 == last) prog[w] = last;
            }
            if (__builtin_expect(have == c && !done, 0)) {
                if (t0 == 0) t0 = __builtin_amdgcn_s_memrealtime();
                if (__builtin_amdgcn_s_memrealtime() - t0 > DEADLINE_TICKS ||
                    __hip_atomic_load(tmo, __ATOMIC_RELAXED, __HIP_MEMORY_SCOPE_AGENT) == call) {
                    __hip_atomic_store(tmo, call, __ATOMIC_RELAXED, __HIP_MEMORY_SCOPE_AGENT);
                    bad = __builtin_nanf("");
                }
                __builtin_amdgcn_s_waitcnt(0x0F70);
            }
            if (done) ++c;
            SB();
            have = cn;
        };
#pragma unroll 1
        while (c <= last) {
#pragma unroll 1
            while (have != last || c != last) pass(BoolC<true>{}, have != c ? c : c + 1);
            pass(BoolC<false>{}, last + 1u);
        }
    }
    bool ok = true;
    const int fsrc = (int)((db >> 20) & 15) - 1, psl = (int)((db >> 24) & 63) - 1;
    if (fsrc >= 0 && fsrc != rank && w == 0) {
        const int qq = (int)((Q_OF_TILE >> (4 * xj)) & 7u);
        const int qs = qq == 7 ? ((rank - fsrc - 1) & 7) : qq;
        const int words = (int)((p.words >> (4 * qs)) & 15u) * NAUX;
        if (words > 0) ok = wait_words<1>((gu32*)(myws + FLAG_OFF + (fsrc * 8 + xj) * 128) + (lane < words ? lane : 0), call, tmo);
        STAMP(7);
    }
    if (psl >= 0) {
        const int b2 = psl * 8 + xj;
        const unsigned d2 = p.slot[psl].a;
        const int tc2 = (int)(col >> (8 * (d2 & 7))) & 63;
        const int src = (2 * rank - (tc2 >> 3)) & 7;
        const __amdgpu_buffer_rsrc_t rm = __builtin_amdgcn_make_buffer_rsrc(p.out[rank], 0, OUT_BYTES, 0x00020000);
        const unsigned pv = (unsigned)((lane >> 3) * NS + (tc2 & 7) * 32 + (lane & 7) * 4) * 4u;
        ok = wait_words<1>((gu32*)(myws + RDY_OFF + b2 * 128) + 2 + (lane & 1), call, tmo) && ok;
        const unsigned long long t0 = __builtin_amdgcn_s_memrealtime();
#pragma unroll 1
        for (int c = w; c < (int)((d2 >> 17) & 15); c += NCW) {
            const unsigned ps = (unsigned)((src * MS + (int)(((d2 >> 7) & 63) + c * ((d2 >> 13) & 15)) * 16) * NS) * 4u;
            for (unsigned it = 1; ok; ++it) {
                asm volatile("" ::: "memory");
                const u32x4 v0 = __builtin_amdgcn_raw_buffer_load_b128(rm, pv, ps, AUX_SYS);
                const u32x4 v1 = __builtin_amdgcn_raw_buffer_load_b128(rm, pv, ps + 8u * NS * 4u, AUX_SYS);
                const unsigned mx = max(max(max(v0[0], v0[1]), max(v0[2], v0[3])), max(max(v1[0], v1[1]), max(v1[2], v1[3])));
                if (!__any(mx == MARK)) break;
                if ((it & 63u) == 0u &&
                    (__hip_atomic_load(tmo, __ATOMIC_RELAXED, __HIP_MEMORY_SCOPE_AGENT) == call ||
                     __builtin_amdgcn_s_memrealtime() - t0 > DEADLINE_TICKS)) {
                    __hip_atomic_store(tmo, call, __ATOMIC_RELAXED, __HIP_MEMORY_SCOPE_AGENT);
                    ok = false;
                }
                __builtin_amdgcn_s_sleep(2);
            }
        }
        STAMP(7);
    }
    if (!ok && lane == 0)
        __hip_atomic_store((gu32*)p.out[rank] + (size_t)rank * MS * NS, 0x7fc00000u, __ATOMIC_RELAXED, __HIP_MEMORY_SCOPE_SYSTEM);
    STAMP_FLUSH();
}

struct Config {
    int ncu[7];
    int self[7];
    int help[7];
    int mark[7];
    int defer[7];
    int imm;
    int fch, nfwd, fn;
    int ohelp;
    int pxn, pxwho;
    int flip;
    int skip;
};

static bool build(const Config& c, int rank, Params& p) {
    memset(p.slot, 0, sizeof p.slot);
    const int rch = 16 - c.fch, och = 16 + 7 * c.fch, nf = c.fch > 0 ? c.nfwd : 0;
    int nrem = 0;
    for (int q = 0; q < 7; ++q) nrem += c.ncu[q];
    const int no = 32 - nrem, np = no - nf, pch = och - nf * c.fn;
    if (np < 1 || c.fch > MAXF || (nf && ((2 * c.fch + nf - 1) / nf > NBF || c.fn < 1))) return false;
    int qs[32], cis[32], sl = 0;
    for (int ci = 0; ci < 8; ++ci)
        for (int q = 0; q < 7; ++q)
            if (ci < c.ncu[q] && !c.help[q]) { qs[sl] = q; cis[sl++] = ci; }
    const int nfirst = sl;
    for (int ci = 0; ci < 8; ++ci)
        for (int q = 0; q < 7; ++q)
            if (ci < c.ncu[q] && c.help[q]) { qs[sl] = q; cis[sl++] = ci; }
    for (int ci = 0; ci < no; ++ci) { qs[sl] = 7; cis[sl++] = ci; }
    unsigned short wl[2][512];
    int nwl[2] = {0, 0};
    for (int s = 0; s < 8; ++s)
        for (int g = 0; g < 8; ++g)
            for (int q = 0; q < 7; ++q)
                if (s >= c.self[q] && !(c.self[q] == 0 && s * 8 + g < c.skip))
                    wl[c.help[q]][nwl[c.help[q]]++] = (unsigned short)(q << 6 | (((s ^ ((c.flip && (q & 1)) ? 4 : 0)) * 8 + g)));
    int nh[32] = {}, zn = 0, nmark = 0;
    const int nconv = np - c.ohelp;
    if (nconv < 1) return false;
    int hs[32], nhelp = 0;
    for (int s = nfirst; s < nrem; ++s) hs[nhelp++] = s;
    for (int s = nrem + nconv; s < nrem + np; ++s) hs[nhelp++] = s;
    for (int s = 0; s < 32; ++s) memset(p.slot[s].h, 0xff, sizeof p.slot[s].h);
    auto add = [&](int s, unsigned short e) {
        if (nh[s] > 0 && nh[s] < MAXH && ((p.slot[s].h[nh[s] - 1] ^ e) & 0x8000)) nh[s] = (nh[s] + 7) / 8 * 8;
        if (nh[s] < MAXH) p.slot[s].h[nh[s]++] = e;
    };
    if (c.pxwho == 0)
        for (int u = 0; u < c.pxn * 16; ++u) add(nrem + u % nconv, (unsigned short)(0x8000 | u));
    for (int i = 0; i < nwl[0] && nhelp > 0; ++i) add(hs[(i + i / nhelp) % nhelp], wl[0][i]);
    if (c.pxwho == 1)
        for (int u = 0; u < c.pxn * 16 && nhelp > 0; ++u) add(hs[u % nhelp], (unsigned short)(0x8000 | u));
    for (int i = 0; i < nwl[1]; ++i) add(nrem + (i + i / np) % np, wl[1][i]);
    p.words = 0;
    for (int q = 0; q < 7; ++q) p.words |= (unsigned)(c.mark[q] ? 0 : c.ncu[q]) << (4 * q);
    for (int s = 0; s < 32; ++s) {
        const int q = qs[s], ci = cis[s];
        const bool own = q == 7, fwd = own && ci >= np, marked = !own && c.mark[q];
        const int cs = own ? (fwd ? nf : np) : c.ncu[q];
        const int c0 = fwd ? pch + ci - np : ci;
        const int n = fwd ? c.fn : ((own ? pch : rch) - ci + cs - 1) / cs;
        if (n < 1 || n > MAXCH) return false;
        Slot& d = p.slot[s];
        d.a = (unsigned)(q | ci << 3 | c0 << 7 | cs << 13 | n << 17 | (int)(own && (fwd || ci < nconv)) << 21 | (int)fwd << 22 | (int)marked << 23 | (nh[s] + 7) / 8 << 24);
        if (own) {
            if (!fwd && (2 * rch + nconv - 1) / nconv > NB) return false;
            d.b = (unsigned)((fwd ? 2 * rch + ci - np : ci) | (fwd ? nf : nconv) << 5 | (fwd ? 32 : 2 * rch) << 9);
        }
        if (s < nfirst && !marked) d.b |= (unsigned)(++zn) << 15;
        if (s < 8) d.b |= (unsigned)(s + 1) << 20;
        if (marked) p.slot[(nfirst - 1 - nmark++ + 32) % 32].b |= (unsigned)(s + 1) << 24;
        const int self = own ? 8 : c.self[q], m = own ? np : c.ncu[q], fl = (!own && c.flip && (q & 1)) ? 4 : 0;
        for (int t = 0; t < 8; ++t)
            d.ord |= (unsigned)(((t < self ? (t + ci * self / m) % self : t) ^ fl) << (4 * t));
        for (int t = 0; t < n; ++t) {
            int r = t;
            if (!own && c.defer[q] == 1 && (ci & 1) && t == 0 && n > 1) r = 1;
            if (!own && c.defer[q] == 2 && t < n - 2 && t >= n - 4 && !(t == 0 && ci >= m - c.imm)) r = n - 2;
            d.rel |= 1u << (4 * r + (r - t));
        }
    }
    if (zn == 0 || nmark > nfirst || nfirst < 8) return false;
    p.cfg = (unsigned)(rch | zn << 5);
    for (int xj = 0; xj < 8; ++xj) {
        p.col[xj] = (unsigned long long)(rank * 8 + xj) << 56;
        for (int q = 0; q < 7; ++q)
            p.col[xj] |= (unsigned long long)(((rank + 1 + (q + xj) % 7) & 7) * 8 + (int)((TILE_OF_Q >> (4 * (xj == 7 ? 7 : q))) & 7u)) << (8 * q);
    }
    return true;
}

extern "C" void kernel_launch(int rank, int nranks, ncclComm_t comm,
                              hipStream_t stream,
                              void* const* d_in, const int* in_sizes, int n_in,
                              void* d_out, int out_size,
                              const int* in_dtypes, int out_dtype,
                              void* const* d_out_all,
                              void* const* d_ws_all, size_t ws_bytes) {
    if (nranks != NR || (unsigned)rank >= (unsigned)NR || n_in != 2 || in_sizes[0] != MS * KK ||
        in_sizes[1] != KK * NN || out_size != NR * MS * NS || ws_bytes < WS_NEED) {
        fprintf(stderr, "gemm_a2a: unexpected problem shape\n");
        return;
    }
    Config c = {{4, 4, 4, 4, 4, 4, 4}, {4, 4, 4, 4, 4, 8, 8}, {0, 0, 0, 0, 0, 1, 1}, {0, 0, 0, 0, 0, 1, 1}, {0, 0, 0, 0, 0, 2, 2},
                2, 0, 0, 0, 0, 4, 0, 1, 4};
    Params p;
    memset(&p, 0, sizeof p);
    if (!build(c, rank, p)) {
        fprintf(stderr, "gemm_a2a: bad schedule\n");
        return;
    }
    p.x = (const float*)d_in[0];
    p.w = (const float*)d_in[1];
    for (int i = 0; i < NR; ++i) {
        p.out[i] = d_out_all[i];
        p.ws[i] = d_ws_all[i];
    }
    p.myws = d_ws_all[rank];
    p.rank = rank;
    hipLaunchKernelGGL(gemm_a2a, dim3(NWG), dim3(NTHR), 0, stream, p);
    HIP_CHECK(hipGetLastError());
}
```
